```python
import numpy as np

kernel_name = "gpu_gemm_rs_m4096_k4096_n4096_bf16_q8"


ABSMAX_RTOL = 0.02

COMPUTE_DTYPE = "bfloat16"
EPILOGUE = "quant_int8"
IN_QUANT = None
OUT_QUANT = "int8"
IN_DTYPES = {"x": "bfloat16", "w_mat": "bfloat16"}
WORLD_SIZE = 8

def _to_bf16(x):
    u = np.ascontiguousarray(x, dtype=np.float32).view(np.uint32)
    return ((u + 0x7FFF + ((u >> 16) & 1)) & 0xFFFF0000).view(np.float32)

def setup_inputs():

    x = _to_bf16(np.random.randn(4096, 4096).astype(np.float32))
    w_mat = _to_bf16((np.random.randn(4096, 4096) / np.sqrt(4096)).astype(np.float32))
    return {"x": x, "w_mat": w_mat}

def _quant_dequant(y, amax):
    scale = amax / 127.0
    q = np.clip(np.round(y / scale), -127.0, 127.0)
    return (q * scale).astype(np.float32)
def _epilogue(y):
    return _quant_dequant(y, np.abs(y).max())

def reference(x, w_mat):
    y = (x @ w_mat).astype(np.float32)
    return _epilogue(y)
```

```cpp
#include <hip/hip_runtime.h>
#include <rccl/rccl.h>
#include <cstdio>
#include <cstdint>
#include <cstring>

#define HIP_CHECK(expr) do { hipError_t _e = (expr); if (_e != hipSuccess) { \
    fprintf(stderr, "HIP error %s at %s:%d\n", hipGetErrorString(_e), __FILE__, __LINE__); \
    return; } } while (0)

namespace {

typedef unsigned int u32;
typedef unsigned long long u64;
typedef u32 u32x4 __attribute__((ext_vector_type(4)));
typedef short s16x4 __attribute__((ext_vector_type(4)));
typedef __bf16 bf16x8 __attribute__((ext_vector_type(8)));
typedef float f32x4 __attribute__((ext_vector_type(4)));
#define LAS __attribute__((address_space(3)))

constexpr int NR = 8;
constexpr int MM = 4096, NN = 4096;
constexpr int KS = 512;
constexpr int RS = 512;
constexpr int NWG = 256, NT = 512;
constexpr int NPC = 4;
constexpr int MAXJ = 48;
constexpr u32 CHUNK = 65536;
constexpr int AUX_WT = 17;
constexpr u64 DEADLINE = 30000000ull;

constexpr u32 OFF_CNT  = 0;
constexpr u32 OFF_TMO  = 4096;
constexpr u32 OFF_LG   = 8192;
constexpr u32 OFF_GG   = 12288;
constexpr u32 OFF_FL   = 16384;
constexpr u32 OFF_DBG  = 131072;
constexpr u32 OFF_CH   = 1u << 20;
constexpr u32 ID_XS = 0;
constexpr u32 ID_WS = 96;
constexpr u32 ID_PS = 192;
constexpr u32 ID_OB = 320;
constexpr u32 N_ID = 448;
constexpr u32 WS_NEED = OFF_CH + N_ID * CHUNK;
static_assert(OFF_FL + N_ID * 64 <= OFF_DBG, "flag area");

constexpr u32 LDS_CTL = 131072;
constexpr u32 LDS_BYTES = LDS_CTL + 256;

struct Params {
  const char* x;
  const char* w;
  char* out;
  char* ws[NR];
  int rank;
  u32 njobs[NR];
  u32 jobs[NR][MAXJ / 4];
};

typedef __amdgpu_buffer_rsrc_t rsrc_t;
__device__ __forceinline__ rsrc_t mk_rsrc(const void* p, u32 bytes) {
  return __builtin_amdgcn_make_buffer_rsrc(const_cast<void*>(p), 0, bytes, 0x00020000);
}
__device__ __forceinline__ u32 ld_sys(const char* p) {
  return __hip_atomic_load((const u32*)p, __ATOMIC_RELAXED, __HIP_MEMORY_SCOPE_SYSTEM);
}
__device__ __forceinline__ void st_sys(char* p, u32 v) {
  __hip_atomic_store((u32*)p, v, __ATOMIC_RELAXED, __HIP_MEMORY_SCOPE_SYSTEM);
}
__device__ __forceinline__ void st_sys64(char* p, u64 v) {
  __hip_atomic_store((u64*)p, v, __ATOMIC_RELAXED, __HIP_MEMORY_SCOPE_SYSTEM);
}
__device__ __forceinline__ u32 cvt_pk_bf16(float lo, float hi) {
  u32 r; asm("v_cvt_pk_bf16_f32 %0, %1, %2" : "=v"(r) : "v"(lo), "v"(hi)); return r;
}
__device__ __forceinline__ u32 rfl(u32 v) { return __builtin_amdgcn_readfirstlane(v); }
__device__ __forceinline__ u32x4 mk_desc(const void* p, u32 bytes) {
  const u64 a = (u64)p;
  return u32x4{rfl((u32)a), rfl((u32)(a >> 32) & 0xffffu), bytes, 0x00020000u};
}
__device__ __forceinline__ void dma16(u32x4 desc, u32 voff, u32 soff, u32 ldsaddr) {
  u32 keep;
  soff = rfl(soff);
  ldsaddr = rfl(ldsaddr);
  asm volatile("s_mov_b32 %0, m0\n\ts_mov_b32 m0, %4\n\ts_nop 2\n\tbuffer_load_dwordx4 %1, %2, %3 offen sc0 sc1 lds\n\ts_mov_b32 m0, %0"
               : "=&s"(keep) : "v"(voff), "s"(desc), "s"(soff), "s"(ldsaddr) : "memory");
}
#define WAIT_VM(n) asm volatile("s_waitcnt vmcnt(" #n ")" ::: "memory")
#define WAIT_LGKM0() asm volatile("s_waitcnt lgkmcnt(0)" ::: "memory")
#define BARRIER() do { asm volatile("" ::: "memory"); __builtin_amdgcn_s_barrier(); asm volatile("" ::: "memory"); } while (0)

__device__ __forceinline__ bool give_up(char* ws, u32 seq, u64 t0, u32& polls) {
  if ((++polls & 63u) != 0u) return false;
  const u32 t = rfl(__hip_atomic_load((const u32*)(ws + OFF_TMO), __ATOMIC_RELAXED, __HIP_MEMORY_SCOPE_AGENT));
  if (t == seq) return true;
  if (__builtin_amdgcn_s_memrealtime() - t0 > DEADLINE) {
    __hip_atomic_store((u32*)(ws + OFF_TMO), seq, __ATOMIC_RELAXED, __HIP_MEMORY_SCOPE_AGENT);
    return true;
  }
  return false;
}

#define STAMP_T(t, i) do {} while (0)
#define STAMP(i) STAMP_T(0, i)

__host__ __device__ __forceinline__ int blk_owner(int r, int j) { return j < 2 ? (r ^ 1) : (j < 4 ? r : (r ^ j)); }
__host__ __device__ __forceinline__ int blk_rb(int r, int j) { return j < 4 ? (j & 1) : ((r >> 1) & 1); }
__host__ __device__ __forceinline__ int khalf(int s, int r) { return (r & 1) ^ (((r >> 1) & 1) & (s & 1)); }
__host__ __device__ __forceinline__ u32 ps_id(int r, int j, int c) {
  return ID_PS + (u32)((((j < 4 ? 0 : 1) * 2 + blk_rb(r, j)) * 16 + c) * 2 + (r & 1));
}

__device__ __forceinline__ void push_role(const Params& p, int d, int cu, u32 seq, char* smem) {
  const int r = p.rank, dst = (r ^ d) & (NR - 1), tid = threadIdx.x, lane = tid & 63;
  const int wv = rfl(tid >> 6);
  char* ws = p.ws[r];
  LAS char* lds = (LAS char*)smem;
  volatile LAS u32* ctl = (volatile LAS u32*)(lds + LDS_CTL);
  if (tid == 0) { ctl[0] = 0; ctl[1] = 0; }
  __syncthreads();
  if (wv > 1) return;
  const u32 nj = p.njobs[d];
  u32 jw = 0;
#pragma unroll
  for (int i = 0; i < MAXJ / 4; ++i) jw = (lane >> 2) == i ? p.jobs[d][i] : jw;
  const u32 job = (jw >> ((lane & 3) * 8)) & 0xffu;
  const u32 jc = job & 15u;
  const u64 t0 = __builtin_amdgcn_s_memrealtime();
  u32 polls = 0;
  constexpr u32 NSLOT = 8;
  if (wv == 0) {
    const u32x4 rx = mk_desc(p.x, MM * KS * 2), rw = mk_desc(p.w, KS * NN * 2), rs = mk_desc(ws, WS_NEED);
    const int h = khalf(r, dst);
    const u32 sid = job < 96 ? ID_WS + (job - 32) + 32 : ID_OB + (d == 1 ? ((job - 96) >> 4) : (u32)d) * 16 + jc;
    u32 so;
    if (job < 16) so = (u32)(blk_owner(dst, job >> 1) * RS + blk_rb(dst, job >> 1) * 256 + (job & 1) * 128) * 1024;
    else if (job < 32) so = jc * 256;
    else so = OFF_CH + sid * CHUNK;
    const u32 dep = OFF_FL + (job >= 32 && (u32)lane < nj ? sid : 0u) * 64 + cu * 4;
    const u32 vx = (u32)(cu * 32 + (lane & 15)) * 1024 + (lane >> 4) * 16 + h * 512;
    const u32 vw = (u32)(h * 256 + cu * 64 + (lane >> 4)) * 8192 + (lane & 15) * 16 + ((dst >> 2) & 1) * 4096;
    const u32 vl = (u32)(cu * 16384 + lane * 16);
    u64 ready = __ballot(job < 32);
    u32 next = 0;
    while (next < nj) {
      const bool rdy = (ready >> next) & 1;
      if (rdy && next - rfl(ctl[1]) < NSLOT) {
        const u32 jb = __builtin_amdgcn_readlane(job, next), s0 = __builtin_amdgcn_readlane(so, next);
        const u32 la = (u32)(size_t)lds + (next % NSLOT) * 16384;
        if (jb < 16) {
#pragma unroll
          for (int i = 0; i < 16; ++i) dma16(rx, vx, s0 + (i >> 2) * 128 + ((i >> 1) & 1) * 16384 + (i & 1) * 64, la + i * 1024);
        } else if (jb < 32) {
#pragma unroll
          for (int i = 0; i < 16; ++i) dma16(rw, vw, s0 + i * 4 * 8192, la + i * 1024);
        } else {
#pragma unroll
          for (int i = 0; i < 16; ++i) dma16(rs, vl, s0 + i * 1024, la + i * 1024);
        }
        ++next;
        WAIT_VM(32);
        if (next > 2) ctl[0] = next - 2;
        continue;
      }
      const u32 v = ld_sys(ws + dep);
      WAIT_VM(0);
      ctl[0] = next;
      ready |= __ballot(v >= seq);
      if (give_up(ws, seq, t0, polls)) ready = ~0ull;
      if (rdy) { while (next - rfl(ctl[1]) >= NSLOT) __builtin_amdgcn_s_sleep(1); }
      else __builtin_amdgcn_s_sleep(1);
    }
    WAIT_VM(0);
    ctl[0] = nj;
  } else {
    char* dws = p.ws[dst];
    const rsrc_t rd = mk_rsrc(dws, WS_NEED);
    u32 did;
    if (job < 16) did = ID_XS + (u32)(d - 2) * 16 + job;
    else if (job < 32) did = ID_WS + (u32)(d - 2) * 16 + jc;
    else if (job < 96) did = ID_WS + (((((job - 32) >> 4) + 4) ^ (u32)d) - 2) * 16 + jc;
    else did = ps_id(r, d == 1 ? (int)((job - 96) >> 4) : d, (int)jc);
    u32 pend = ~0u;
    for (u32 k = 0; k < nj; ++k) {
      if (rfl(ctl[0]) <= k) {
        if (pend != ~0u) { WAIT_VM(0); if (lane == 0) st_sys(dws + pend, seq); pend = ~0u; }
        while (rfl(ctl[0]) <= k) __builtin_amdgcn_s_sleep(1);
      }
      const u32 id = __builtin_amdgcn_readlane(did, k);
      const LAS u32x4* src = (const LAS u32x4*)(lds + (k % NSLOT) * 16384) + lane;
      u32x4 b[16];
#pragma unroll
      for (int i = 0; i < 16; ++i) b[i] = src[i * 64];
      WAIT_LGKM0();
      ctl[1] = k + 1;
      const u32 so = OFF_CH + id * CHUNK + cu * 16384;
#pragma unroll
      for (int i = 0; i < 16; ++i) {
        WAIT_VM(15);
        if (i == 15 && pend != ~0u) { if (lane == 0) st_sys(dws + pend, seq); }
        __builtin_amdgcn_raw_buffer_store_b128(b[i], rd, lane * 16 + (i & 3) * 1024, so + (i >> 2) * 4096, AUX_WT);
      }
      pend = OFF_FL + id * 64 + cu * 4;
      if ((k & 1) == 1 && k < 60) STAMP_T(64, 2 + (k >> 1));
    }
    WAIT_VM(0);
    if (pend != ~0u && lane == 0) st_sys(dws + pend, seq);
    STAMP_T(64, 1);
  }
}

struct Quad { u32x4 dx, dw; u32 sx, sw, x_step, x_mb, x_kk, w_step, w_i, vx, vw; };

__device__ __forceinline__ void xload(bf16x8& dst, u32x4 desc, u32 voff, u32 soff) {
  soff = rfl(soff);
  asm volatile("s_nop 4\n\tbuffer_load_dwordx4 %0, %1, %2, %3 offen sc0 sc1" : "+v"(dst) : "v"(voff), "s"(desc), "s"(soff) : "memory");
}
#define TIE_X(X) asm volatile("" : "+v"(X[0]), "+v"(X[1]), "+v"(X[2]), "+v"(X[3]) :: "memory")

__device__ __forceinline__ void gemm_tile(const Params& p, int j, int c, int rt, u32 seq, char* smem, int sb) {
  const int r = p.rank, tid = threadIdx.x, lane = tid & 63;
  const int wv8 = rfl(tid >> 6), grp = wv8 >> 2, wv = wv8 & 3;
  char* ws = p.ws[r];
  const u32x4 rx = mk_desc(p.x, MM * KS * 2), rw = mk_desc(p.w, KS * NN * 2), rs = mk_desc(ws, WS_NEED);
  const int fr = lane & 15, fq = lane >> 4;
  const u32 krow = wv * 16 + fq, wch = (u32)(fr ^ fq);
  LAS char* lds = (LAS char*)smem + grp * 65536;
  const u32 ldsw = (u32)(size_t)lds + wv * 4096;

  auto quad = [&](int pr) -> Quad {
    Quad o;
    if (pr == 0) {
      o.dx = rx; o.dw = rw;
      o.sx = (u32)(blk_owner(r, j) * RS + blk_rb(r, j) * 256 + rt * 128) * 1024 + grp * 512;
      o.x_step = 128; o.x_mb = 16384; o.x_kk = 64;
      o.sw = (u32)(((r >> 2) & 1) * 2048 + c * 128) * 2 + (u32)grp * 256 * 8192; o.w_step = 64 * 8192; o.w_i = 4 * 8192;
      o.vx = (u32)(wv * 32 + fr) * 1024 + fq * 16; o.vw = krow * 8192 + wch * 16;
    } else {
      const u32 dm2 = (u32)((pr == 3 ? 0 : pr * 2) + grp) * 16;
      o.dx = rs; o.dw = rs;
      o.sx = OFF_CH + (ID_XS + dm2 + j * 2 + rt) * CHUNK; o.x_step = 4096; o.x_mb = 2048; o.x_kk = 1024;
      o.sw = OFF_CH + (ID_WS + dm2 + c) * CHUNK; o.w_step = 16384; o.w_i = 1024;
      o.vx = (u32)(wv * 16384 + lane * 16); o.vw = krow * 256 + wch * 16;
    }
    o.sx = rfl(o.sx); o.sw = rfl(o.sw);
    return o;
  };
  bf16x8 xr[4][4];
#pragma unroll
  for (int a = 0; a < 4; ++a)
#pragma unroll
    for (int b = 0; b < 4; ++b) xr[a][b] = bf16x8{};
  auto issue = [&](const Quad& o, int st, bf16x8* x) {
#pragma unroll
    for (int i = 0; i < 4; ++i) {
      dma16(o.dw, o.vw ^ ((i >> 1) * 128), o.sw + st * o.w_step + i * o.w_i, ldsw + st * 16384 + i * 1024);
      xload(x[i], o.dx, o.vx, o.sx + st * o.x_step + (i >> 1) * o.x_mb + (i & 1) * o.x_kk);
    }
  };
  const int sg = lane >> 3, sdm2 = sg < 4 ? sg + 2 : (sg < 6 ? sg - 4 : 0);
  const u32 pgo = OFF_FL + ((lane & 4) ? ID_WS + sdm2 * 16 + c : ID_XS + sdm2 * 16 + j * 2 + rt) * 64 + (lane & 3) * 4;
  u32 pv = 0;
  auto pairs_ready = [&](u32 v) -> u32 {
    return (u32)__builtin_ctzll(~(__ballot(v >= seq) & 0xffffffffffffull)) >> 4;
  };
  volatile LAS u32* kn = (volatile LAS u32*)((LAS char*)smem + LDS_CTL);

  const int qq = fr >> 2, pp = lane & 3;
  const u32 wsw = (u32)(qq | ((fq & 1) << 3));
  u32 wa[4];
#pragma unroll
  for (int t = 0; t < 4; ++t) {
    const u32 tt = (u32)(t ^ (pp & 1));
    wa[t] = (u32)(8 * fq + qq) * 256 + ((((u32)(pp * 2) + (tt >> 1)) ^ wsw) * 16) + (tt & 1) * 8;
  }
  f32x4 acc[2][8];
#pragma unroll
  for (int a = 0; a < 2; ++a)
#pragma unroll
    for (int b = 0; b < 8; ++b) acc[a][b] = f32x4{0.f, 0.f, 0.f, 0.f};
  bf16x8 wf[2][8];
  auto wread = [&](int st) {
#pragma unroll
    for (int kk = 0; kk < 2; ++kk)
#pragma unroll
      for (int t = 0; t < 8; ++t) {
        const LAS char* a = lds + (wa[t & 3] ^ ((t >> 2) * 128)) + st * 16384 + kk * 8192;
        const s16x4 lo = __builtin_amdgcn_ds_read_tr16_b64_v4i16((LAS s16x4*)a);
        const s16x4 hi = __builtin_amdgcn_ds_read_tr16_b64_v4i16((LAS s16x4*)(a + 1024));
        wf[kk][t] = __builtin_bit_cast(bf16x8, __builtin_shufflevector(lo, hi, 0, 1, 2, 3, 4, 5, 6, 7));
      }
    __builtin_amdgcn_sched_barrier(0);
  };
  auto mma = [&](const bf16x8* x) {
    __builtin_amdgcn_sched_barrier(0);
    __builtin_amdgcn_s_setprio(1);
#pragma unroll
    for (int kk = 0; kk < 2; ++kk)
#pragma unroll
      for (int t = 0; t < 8; ++t) {
        acc[0][t] = __builtin_amdgcn_mfma_f32_16x16x32_bf16(wf[kk][t], x[kk], acc[0][t], 0, 0, 0);
        acc[1][t] = __builtin_amdgcn_mfma_f32_16x16x32_bf16(wf[kk][t], x[2 + kk], acc[1][t], 0, 0, 0);
      }
    __builtin_amdgcn_s_setprio(0);
    __builtin_amdgcn_sched_barrier(0);
  };

  u32 known = 0, polls = 0;
  const u64 t0 = __builtin_amdgcn_s_memrealtime();
  __builtin_amdgcn_s_waitcnt(0xC07F);
  Quad cur = quad(0);
  bool restart = true;
  for (int pr = 0; pr < 4; ++pr) {
    const Quad nxt = quad(pr + 1);
    if (restart) {
      if (wv8 == 0) asm volatile("buffer_load_dword %0, %1, %2, 0 offen sc0 sc1" : "+v"(pv) : "v"(pgo), "s"(rs) : "memory");
      issue(cur, 0, xr[0]); issue(cur, 1, xr[1]); issue(cur, 2, xr[2]);
      WAIT_VM(16);
      BARRIER();
      if (grp == 1) BARRIER();
    }
    STAMP(sb + pr);
    const bool ask = pr < 3 && known <= (u32)pr;
    asm volatile("" : "+v"(pv) :: "memory");
    if (ask && wv8 == 0) *kn = pairs_ready(pv);
    wread(0); issue(cur, 3, xr[3]); WAIT_LGKM0(); BARRIER();
    TIE_X(xr[0]); mma(xr[0]); WAIT_VM(16); BARRIER();
    if (ask) known = rfl(*kn);
    const bool go = pr < 3 && known > (u32)pr;
    wread(1); if (go) issue(nxt, 0, xr[0]); WAIT_LGKM0(); BARRIER();
    TIE_X(xr[1]); mma(xr[1]); if (go) WAIT_VM(16); else WAIT_VM(8); BARRIER();
    wread(2);
    if (go) {
      if (wv8 == 0) asm volatile("buffer_load_dword %0, %1, %2, 0 offen sc0 sc1" : "+v"(pv) : "v"(pgo), "s"(rs) : "memory");
      issue(nxt, 1, xr[1]);
    }
    WAIT_LGKM0(); BARRIER();
    TIE_X(xr[2]); mma(xr[2]); if (go) WAIT_VM(16); else WAIT_VM(0); BARRIER();
    wread(3); if (go) issue(nxt, 2, xr[2]); WAIT_LGKM0(); BARRIER();
    TIE_X(xr[3]); mma(xr[3]); if (go) WAIT_VM(16); BARRIER();
    restart = !go;
    if (!go) {
      if (grp == 0) BARRIER();
      if (pr < 3) {
        if (wv8 == 0) {
          u32 k;
          while (true) {
            k = pairs_ready(ld_sys(ws + pgo));
            if (k > (u32)pr) break;
            if (give_up(ws, seq, t0, polls)) { k = 3; break; }
            __builtin_amdgcn_s_sleep(2);
          }
          *kn = k;
          WAIT_LGKM0();
        }
        BARRIER();
        known = rfl(*kn);
      }
    }
    cur = nxt;
  }
  STAMP(sb + 4);
  LAS f32x4* ex = (LAS f32x4*)smem;
#pragma unroll
  for (int t = 0; t < 8; ++t) ex[((grp * 4 + wv) * 8 + t) * 64 + lane] = grp ? acc[0][t] : acc[1][t];
  WAIT_LGKM0();
  BARRIER();
  const u32 id = (j & 6) == 2 ? ps_id(r, j, c) : ID_OB + (u32)j * 16 + c;
  const rsrc_t rd = mk_rsrc(ws, WS_NEED);
  const u32 so = OFF_CH + id * CHUNK + rt * 32768 + wv * 8192 + grp * 4096;
#pragma unroll
  for (int hf = 0; hf < 4; ++hf) {
    const f32x4 a = (grp ? acc[1][2 * hf] : acc[0][2 * hf]) + ex[(((grp ^ 1) * 4 + wv) * 8 + 2 * hf) * 64 + lane];
    const f32x4 b = (grp ? acc[1][2 * hf + 1] : acc[0][2 * hf + 1]) + ex[(((grp ^ 1) * 4 + wv) * 8 + 2 * hf + 1) * 64 + lane];
    const u32x4 v = {cvt_pk_bf16(a[0], a[1]), cvt_pk_bf16(a[2], a[3]), cvt_pk_bf16(b[0], b[1]), cvt_pk_bf16(b[2], b[3])};
    __builtin_amdgcn_raw_buffer_store_b128(v, rd, lane * 16 + hf * 1024, so, AUX_WT);
  }
  WAIT_VM(0);
  WAIT_LGKM0();
  BARRIER();
  if (tid < 2) st_sys(ws + OFF_FL + id * 64 + (rt * 2 + tid) * 4, seq);
  STAMP(sb + 5);
}

__device__ __forceinline__ float as_f(u32 v) { return __builtin_bit_cast(float, v); }
__device__ __forceinline__ float bf_lo(u32 v) { return as_f(v << 16); }
__device__ __forceinline__ float bf_hi(u32 v) { return as_f(v & 0xffff0000u); }
__device__ __forceinline__ float wave_max(float v) {
#pragma unroll
  for (int o = 32; o > 0; o >>= 1) v = fmaxf(v, __shfl_xor(v, o));
  return v;
}

__device__ __forceinline__ void reduce_tile(const Params& p, int type, int rb, int c, int rt, u32 seq, char* smem) {
  const int r = p.rank, tid = threadIdx.x, lane = tid & 63;
  const int wv = rfl(tid >> 6);
  char* ws = p.ws[r];
  const rsrc_t rs = mk_rsrc(ws, WS_NEED);
  const u32 id0 = ID_PS + (u32)(((type * 2 + rb) * 16 + c) * 2);
  const u64 t0 = __builtin_amdgcn_s_memrealtime();
  volatile LAS float* red = (volatile LAS float*)((LAS char*)smem + LDS_CTL + 64);
  u32 polls = 0;
  if (wv == 0) {
    const char* fp = ws + OFF_FL + (id0 + (lane & 1)) * 64 + (rt * 2 + ((lane >> 1) & 1)) * 4;
    while (true) {
      const u32 v = ld_sys(fp);
      if (__all(v >= seq)) break;
      if (give_up(ws, seq, t0, polls)) break;
      __builtin_amdgcn_s_sleep(2);
    }
  }
  BARRIER();
  STAMP(24);
  const u32 so = OFF_CH + id0 * CHUNK + rt * 32768 + wv * 4096;
  u32x4 a[4], b[4];
#pragma unroll
  for (int i = 0; i < 4; ++i) {
    a[i] = __builtin_amdgcn_raw_buffer_load_b128(rs, lane * 16 + i * 1024, so, AUX_WT);
    b[i] = __builtin_amdgcn_raw_buffer_load_b128(rs, lane * 16 + i * 1024, so + CHUNK, AUX_WT);
  }
  float y[4][8];
  float m = 0.f;
#pragma unroll
  for (int i = 0; i < 4; ++i)
#pragma unroll
    for (int e = 0; e < 4; ++e) {
      y[i][2 * e] = bf_lo(a[i][e]) + bf_lo(b[i][e]);
      y[i][2 * e + 1] = bf_hi(a[i][e]) + bf_hi(b[i][e]);
      m = fmaxf(m, fmaxf(fabsf(y[i][2 * e]), fabsf(y[i][2 * e + 1])));
    }
  m = wave_max(m);
  if (lane == 0) red[wv] = m;
  WAIT_LGKM0();
  BARRIER();
  if (wv == 0) {
    m = wave_max(red[lane & 7]);
    const int col4 = rb * 2 + rt;
    char* lg = ws + OFF_LG + col4 * 256;
    if (type == 0 || c != 15) {
      if (lane == 0) st_sys64(lg + (type * 16 + c) * 8, ((u64)seq << 32) | __builtin_bit_cast(u32, m));
    } else {
      const int sl = lane & 31;
      u64 g;
      while (true) {
        g = __hip_atomic_load((const u64*)(lg + sl * 8), __ATOMIC_RELAXED, __HIP_MEMORY_SCOPE_SYSTEM);
        if (__all(sl == 31 || (u32)(g >> 32) >= seq)) break;
        if (give_up(ws, seq, t0, polls)) break;
        __builtin_amdgcn_s_sleep(1);
      }
      m = wave_max(sl == 31 ? m : as_f((u32)g));
      if (lane < NR) st_sys64(p.ws[lane] + OFF_GG + (r * 4 + col4) * 8, ((u64)seq << 32) | __builtin_bit_cast(u32, m));
    }
    STAMP(25);
    u64 g;
    while (true) {
      g = __hip_atomic_load((const u64*)(ws + OFF_GG + (lane & 31) * 8), __ATOMIC_RELAXED, __HIP_MEMORY_SCOPE_SYSTEM);
      if (__all((u32)(g >> 32) >= seq)) break;
      if (give_up(ws, seq, t0, polls)) break;
      __builtin_amdgcn_s_sleep(1);
    }
    m = wave_max(as_f((u32)g));
    const bool bad = __hip_atomic_load((const u32*)(ws + OFF_TMO), __ATOMIC_RELAXED, __HIP_MEMORY_SCOPE_AGENT) == seq;
    if (lane == 0) red[8] = bad ? -1.f : m;
    WAIT_LGKM0();
  }
  BARRIER();
  const float gm = red[8];
  STAMP(26);
  const bool bad = gm < 0.f;
  const float inv = gm > 0.f ? 127.0f / gm : 0.f;
  const int fr = lane & 15, fq = lane >> 4;
  const int colhalf = ((r >> 2) & 1) ^ type;
#pragma unroll
  for (int mi = 0; mi < 2; ++mi) {
    u32 d[4];
#pragma unroll
    for (int t = 0; t < 4; ++t) {
      u32 pk = 0;
#pragma unroll
      for (int e = 0; e < 4; ++e) {
        float q = __builtin_rintf(y[mi * 2 + (t >> 1)][(t & 1) * 4 + e] * inv);
        q = fminf(fmaxf(q, -127.f), 127.f);
        pk |= ((u32)(int)q & 0xffu) << (8 * e);
      }
      d[t] = bad ? 0x80808080u : pk;
    }
    const bool odd = fq & 1;
    const u32x4 v = {odd ? d[1] : d[0], odd ? d[0] : d[1], odd ? d[3] : d[2], odd ? d[2] : d[3]};
    const int row = rb * 256 + rt * 128 + wv * 16 + fr;
    const int col = colhalf * 2048 + c * 128 + mi * 64 + fq * 16;
    *(u32x4*)(p.out + (size_t)row * NN + col) = v;
  }
  STAMP(27);
}

__global__ void __launch_bounds__(NT, 2) fused_kernel(Params p) {
  extern __shared__ __attribute__((aligned(16))) char smem[];
  char* ws = p.ws[p.rank];
  STAMP(0);
  const int j = blockIdx.x & 7, wi = blockIdx.x >> 3;
  u32* cnt = (u32*)(ws + OFF_CNT) + blockIdx.x;
  const u32 seq = rfl(__hip_atomic_load(cnt, __ATOMIC_RELAXED, __HIP_MEMORY_SCOPE_AGENT)) + 1;
  if (j != 0 && wi < NPC) {
    push_role(p, j, wi, seq, smem);
    if (threadIdx.x == 64) __hip_atomic_store(cnt, seq, __ATOMIC_RELAXED, __HIP_MEMORY_SCOPE_AGENT);
    return;
  }
  const int wk = j == 0 ? wi : wi - NPC;
  int j2 = -1, t2 = 0;
  if (j == 0) { if (wk < 8) { j2 = 2 + (wk >> 2); t2 = 28 + (wk & 3); } }
  else if ((j & 6) != 2 && wk < 4) { j2 = j; t2 = 28 + wk; }
  int jj = j, tt = wk;
  for (int it = 0; it < 2; ++it) {
    gemm_tile(p, rfl(jj), rfl(tt >> 1), rfl(tt & 1), seq, smem, 8 + it * 8);
    if (j2 < 0) break;
    jj = j2; tt = t2; j2 = -1;
  }
  int u = -1;
  if ((jj & 6) == 2) reduce_tile(p, 0, jj & 1, tt >> 1, tt & 1, seq, smem);
  else if (j == 0) u = 40 + wk - 8;
  else if (j == 1) { if (wk >= 4) u = 16 + wk - 4; }
  else if (wk >= 4 && wk < 8) u = (wk - 4) * 4 + (j - 4);
  if (u >= 0) reduce_tile(p, 1, (u >> 1) & 1, u >> 2, u & 1, seq, smem);
  if (threadIdx.x == 0) __hip_atomic_store(cnt, seq, __ATOMIC_RELAXED, __HIP_MEMORY_SCOPE_AGENT);
}

void build_jobs(Params& p) {
  for (int d = 0; d < NR; ++d) {
    unsigned char q[MAXJ];
    memset(q, 0, sizeof q);
    int n = 0;
    if (d == 1) {
      for (int c = 0; c < 16; ++c) { q[n++] = 96 + c; q[n++] = 96 + 16 + c; }
    } else if (d >= 2) {
      for (int i = 0; i < 4; ++i) q[n++] = i;
      for (int i = 0; i < 8; ++i) { q[n++] = 16 + i; q[n++] = 8 + i; }
      for (int i = 8; i < 16; ++i) q[n++] = 16 + i;
      for (int i = 4; i < 8; ++i) q[n++] = i;
      if (d >= 4) for (int c = 0; c < 16; ++c) q[n++] = 96 + c;
    }
    p.njobs[d] = n;
    memcpy(p.jobs[d], q, MAXJ);
  }
}

}

extern "C" void kernel_launch(int rank, int nranks, ncclComm_t comm,
                              hipStream_t stream,
                              void* const* d_in, const int* in_sizes, int n_in,
                              void* d_out, int out_size,
                              const int* in_dtypes, int out_dtype,
                              void* const* d_out_all,
                              void* const* d_ws_all, size_t ws_bytes) {
  (void)comm; (void)in_dtypes; (void)out_dtype; (void)d_out_all;
  if (nranks != NR || n_in != 2 || in_sizes[0] != MM * KS || in_sizes[1] != KS * NN || out_size != RS * NN ||
      ws_bytes < WS_NEED || (unsigned)rank >= (unsigned)NR) {
    fprintf(stderr, "unexpected problem shape\n");
    return;
  }
  HIP_CHECK(hipFuncSetAttribute((const void*)fused_kernel, hipFuncAttributeMaxDynamicSharedMemorySize, LDS_BYTES));
  Params p;
  p.x = (const char*)d_in[0];
  p.w = (const char*)d_in[1];
  p.out = (char*)d_out;
  for (int i = 0; i < NR; ++i) p.ws[i] = (char*)d_ws_all[i];
  p.rank = rank;
  build_jobs(p);
  hipLaunchKernelGGL(fused_kernel, dim3(NWG), dim3(NT), LDS_BYTES, stream, p);
  HIP_CHECK(hipGetLastError());
}
```

```cpp
#include <hip/hip_runtime.h>
#include <rccl/rccl.h>
#include <cstdio>
#include <cstdint>
#include <cstring>

#define HIP_CHECK(expr) do { hipError_t _e = (expr); if (_e != hipSuccess) { \
    fprintf(stderr, "HIP error %s at %s:%d\n", hipGetErrorString(_e), __FILE__, __LINE__); \
    return; } } while (0)

namespace {

typedef unsigned int u32;
typedef unsigned long long u64;
typedef u32 u32x4 __attribute__((ext_vector_type(4)));
typedef short s16x4 __attribute__((ext_vector_type(4)));
typedef __bf16 bf16x8 __attribute__((ext_vector_type(8)));
typedef float f32x4 __attribute__((ext_vector_type(4)));
#define LAS __attribute__((address_space(3)))
#define GAS __attribute__((address_space(1)))

constexpr int NR = 8;
constexpr int MM = 4096, NN = 4096;
constexpr int KS = 512;
constexpr int RS = 512;
constexpr int NWG = 256, NT = 512;
constexpr int NPC = 4;
constexpr int MAXU = 96;
constexpr int LOWM = 3;
constexpr int NSEND = 2;
constexpr int REL = 3;
constexpr int GSLEEP = 0;
constexpr int FSLEEP = 1;
constexpr int GNEAR = 12;
constexpr int NEARLY = 3;
constexpr unsigned YDIR = (0xffffu << (NEARLY + 9) | ((1u << NEARLY) - 1)) & 0xffffu;
constexpr unsigned PRE = 0x0005;
constexpr int XEARLY = 1;
constexpr int XRELN = 1;
constexpr int XWIN = 3;
constexpr int XDIR = 2;
constexpr int CODE_LINES = 176;
constexpr int OAUX = 16;
constexpr int PRETOUCH = 3;
constexpr int STARTDELAY = 3;
constexpr u32 CHUNK = 65536;
constexpr int AUX_WT = 17;
constexpr u64 DEADLINE = 30000000ull;

constexpr u32 OFF_CNT  = 0;
constexpr u32 OFF_TMO  = 8192;
constexpr u32 OFF_TK   = 9216;
constexpr u32 OFF_VD   = 12288;
constexpr u32 OFF_GG   = 16384;
constexpr u32 OFF_FL   = 24576;
constexpr u32 OFF_CH   = 1u << 20;
constexpr u32 ID_XS = 0;
constexpr u32 ID_WS = 96;
constexpr u32 ID_PS = 192;
constexpr u32 ID_OB = 320;
constexpr u32 ID_TR = 448;
constexpr u32 N_ID = 480;
constexpr u32 WS_NEED = OFF_CH + N_ID * CHUNK;
static_assert(OFF_FL + N_ID * 64 <= OFF_CH, "flag area");

constexpr u32 LDS_CTL = 131072;
constexpr u32 LDS_BYTES = LDS_CTL + 256;

typedef u32 u32x16 __attribute__((ext_vector_type(16)));
struct Grp { const char* x; const char* w; char* ws; char* dws; u64 first8; u32 rank, link, nunits, open, pad[2]; };

static_assert(sizeof(Grp) == 64, "Grp");
struct Params {
  u32x16 grp[NR];
  const char* x;
  const char* w;
  char* out;
  char* wsd[NR];
  int rank;
  alignas(16) unsigned char units[NR][128];
  u32 plan[NWG];
};

typedef __amdgpu_buffer_rsrc_t rsrc_t;
__device__ __forceinline__ rsrc_t mk_rsrc(const void* p, u32 bytes) {
  return __builtin_amdgcn_make_buffer_rsrc(const_cast<void*>(p), 0, bytes, 0x00020000);
}
__device__ __forceinline__ u32 ld_sys(const char* p) {
  return __hip_atomic_load((const GAS u32*)p, __ATOMIC_RELAXED, __HIP_MEMORY_SCOPE_SYSTEM);
}
__device__ __forceinline__ void st_sys(char* p, u32 v) {
  __hip_atomic_store((GAS u32*)p, v, __ATOMIC_RELAXED, __HIP_MEMORY_SCOPE_SYSTEM);
}
__device__ __forceinline__ void st_sys64(char* p, u64 v) {
  __hip_atomic_store((GAS u64*)p, v, __ATOMIC_RELAXED, __HIP_MEMORY_SCOPE_SYSTEM);
}
__device__ __forceinline__ u32 cvt_pk_bf16(float lo, float hi) {
  u32 r; asm("v_cvt_pk_bf16_f32 %0, %1, %2" : "=v"(r) : "v"(lo), "v"(hi)); return r;
}
__device__ __forceinline__ float as_f(u32 v) { return __builtin_bit_cast(float, v); }
__device__ __forceinline__ float bf_lo(u32 v) { return as_f(v << 16); }
__device__ __forceinline__ float bf_hi(u32 v) { return as_f(v & 0xffff0000u); }
__device__ __forceinline__ u32 rfl(u32 v) { return __builtin_amdgcn_readfirstlane(v); }
__device__ __forceinline__ int fresh_tid() { int t = threadIdx.x; asm volatile("" : "+v"(t)); return t; }
__device__ __forceinline__ u32x4 mk_desc(const void* p, u32 bytes) {
  const u64 a = (u64)p;
  return u32x4{rfl((u32)a), rfl((u32)(a >> 32) & 0xffffu), bytes, 0x00020000u};
}
#define WAIT_VM(n) asm volatile("s_waitcnt vmcnt(" #n ")" ::: "memory")
#define WAIT_LGKM0() asm volatile("s_waitcnt lgkmcnt(0)" ::: "memory")
#define BARRIER() do { asm volatile("" ::: "memory"); __builtin_amdgcn_s_barrier(); asm volatile("" ::: "memory"); } while (0)

__device__ __forceinline__ bool give_up(char* ws, u32 seq, u64 t0, u32& polls) {
  if ((++polls & 63u) != 0u) return false;
  const u32 t = rfl(__hip_atomic_load((const GAS u32*)(ws + OFF_TMO), __ATOMIC_RELAXED, __HIP_MEMORY_SCOPE_AGENT));
  if (t == seq) return true;
  if (__builtin_amdgcn_s_memrealtime() - t0 > DEADLINE) {
    __hip_atomic_store((GAS u32*)(ws + OFF_TMO), seq, __ATOMIC_RELAXED, __HIP_MEMORY_SCOPE_AGENT);
    return true;
  }
  return false;
}


__host__ __device__ __forceinline__ int blk_owner(int r, int j) { return j < 2 ? (r ^ 1) : (j < 4 ? r : (r ^ j)); }
__host__ __device__ __forceinline__ int blk_rb(int r, int j) { return j < 4 ? (j & 1) : ((r >> 1) & 1); }
__host__ __device__ __forceinline__ int khalf(int s, int r) { return (r & 1) ^ (((r >> 1) & 1) & (s & 1)); }
__host__ __device__ __forceinline__ u32 ps_id(int r, int j, int c) {
  return ID_PS + (u32)((((j < 4 ? 0 : 1) * 2 + blk_rb(r, j)) * 16 + c) * 2 + (r & 1));
}

struct Claim { u32 cnt, old; };
__device__ __forceinline__ Claim claim_issue(char* ws, u32 slot) {
  Claim c = {0, 0};
  if (threadIdx.x == NT - 64) {
    c.cnt = __hip_atomic_load((GAS u32*)(ws + OFF_CNT) + blockIdx.x, __ATOMIC_RELAXED, __HIP_MEMORY_SCOPE_AGENT);
    c.old = __hip_atomic_fetch_add((GAS u32*)(ws + OFF_CNT + 4096) + slot, 1u, __ATOMIC_RELAXED, __HIP_MEMORY_SCOPE_AGENT);
  }
  return c;
}
__device__ __forceinline__ u32 claim_resolve(char* ws, Claim c, u32& slot, char* smem) {
  volatile LAS u32* sh = (volatile LAS u32*)((LAS char*)smem + LDS_CTL + 240);
  if (threadIdx.x == NT - 64) {
    u32 sl = slot;
    for (int i = 0; __builtin_expect(c.old != c.cnt, 0) && i < 4 * NWG; ++i) {
      __hip_atomic_fetch_sub((GAS u32*)(ws + OFF_CNT + 4096) + sl, 1u, __ATOMIC_RELAXED, __HIP_MEMORY_SCOPE_AGENT);
      sl = (sl + 1) & (NWG - 1);
      c.old = __hip_atomic_fetch_add((GAS u32*)(ws + OFF_CNT + 4096) + sl, 1u, __ATOMIC_RELAXED, __HIP_MEMORY_SCOPE_AGENT);
    }
    __hip_atomic_store((GAS u32*)(ws + OFF_CNT) + blockIdx.x, c.cnt + 1, __ATOMIC_RELAXED, __HIP_MEMORY_SCOPE_AGENT);
    sh[0] = sl; sh[1] = c.cnt + 1;
  }
  WAIT_LGKM0();
  BARRIER();
  slot = rfl(sh[0]);
  return rfl(sh[1]);
}

__device__ __forceinline__ u32 unit_src(u32 code, int d, int cu) {
  const u32 c = (code >> 1) & 15u, hh = code & 1u, b5 = (code >> 5) & 1u;
  const u32 ob = d == 1 ? (code < 192 ? 5 + 2 * b5 - (c & 1) : b5) : (d >= 4 ? (u32)d : (b5 ? (u32)d ^ 1u : (u32)d - 2));
  return code < 192 && d != 1 ? (ID_WS + (((code - 64) >> 5) + 2) * 16 + c) * 8 + cu * 2 + hh
                              : (ID_OB + ob * 16 + c) * 8 + hh * 4 + cu;
}
__device__ __forceinline__ u32 tile_dst(u32 code, int r, int d) {
  const int c = (code >> 1) & 15, b5 = (code >> 5) & 1;
  if (code < 192) return ID_TR + (u32)(b5 * 16 + c);
  if (d == 1 || d >= 4) return ps_id(r, d == 1 ? b5 : d, c);
  return b5 ? ps_id(r ^ d ^ 1, (d ^ 1) - 2, c) : ID_OB + (u32)d * 16 + c;
}
template <bool SPEC>
__device__ __forceinline__ bool push_role(const Params& p, u32& slot, u32& seq, char* smem) {
  const int g = slot & 7, cu = slot >> 3;
  const u32x16 gv = p.grp[g];
  auto ptr = [&](int i) { return (char*)((u64)gv[i] | (u64)gv[i + 1] << 32); };
  const int r = gv[10], d = gv[11], dst = (r ^ d) & (NR - 1), tid = threadIdx.x, lane = tid & 63;
  const int wv = rfl(tid >> 6);
  char* ws = ptr(4);
  char* dws = ptr(6);
  const u32 n = gv[12];
  const u64 first8 = (u64)ptr(8);
  Claim cl = {0, 0};
  if (SPEC) cl = claim_issue(ws, slot);
  u32 k = wv, code = (u32)(first8 >> (wv * 8)) & 0xffu;
  const bool fixed = d != 1;
  const bool early = fixed && wv < 4;
  volatile LAS u32* st = (volatile LAS u32*)smem;
  if (wv == 6) st[lane] = !fixed ? 0u : (lane == 8 ? 0xffu : (lane < NSEND ? 2u : 0u));
  if (wv & 2) __builtin_amdgcn_s_sleep(2);
  const rsrc_t rx = mk_rsrc(ptr(0), MM * KS * 2), rw = mk_rsrc(ptr(2), KS * NN * 2), rs = mk_rsrc(ws, WS_NEED), rd = mk_rsrc(dws, WS_NEED);
  const int h = khalf(r, dst);
  const u32 vx = (u32)(cu * 32 + (lane & 15)) * 1024 + (lane >> 4) * 16 + h * 512;
  const u32 vw = (u32)(h * 256 + cu * 64 + (lane >> 4)) * 8192 + (lane & 15) * 16 + ((dst >> 2) & 1) * 4096;
  const u32 vl = (u32)lane * 16;
  u32 polls = 0, c0 = 0, c1 = 0, f0 = 0, f1 = 0;
  u64 rdy0 = 0, rdy1 = 0, t0 = 0;
  const u64 val0 = n >= 64 ? ~0ull : (1ull << n) - 1, val1 = n > 64 ? (1ull << (n - 64)) - 1 : 0ull;
  auto confirm = [&]() -> bool {
    if (SPEC) {
      seq = claim_resolve(ws, cl, slot, smem);
      if (slot != (u32)(cu * 8 + g)) { WAIT_VM(0); return false; }
    } else {
      WAIT_LGKM0();
      BARRIER();
    }
    return true;
  };
  auto send = [&](bool first) -> bool {
    if (!(first && early)) st[wv] = k * 4 + 3;
    const u32 c = (code >> 1) & 15u, hh = code & 1u;
    u32x4 b[8];
    u32 du;
    if (code < 32) {
      const u32 so = (u32)(blk_owner(dst, c >> 1) * RS + blk_rb(dst, c >> 1) * 256 + (c & 1) * 128 + hh * 16) * 1024;
#pragma unroll
      for (int q = 0; q < 8; ++q) b[q] = __builtin_amdgcn_raw_buffer_load_b128(rx, vx + q * 64, so, 0);
      du = (ID_XS + (u32)(d - 2) * 16 + c) * 8 + cu * 2 + hh;
    } else if (first || code < 64) {
      const u32 so = c * 256 + hh * 32 * 8192;
#pragma unroll
      for (int q = 0; q < 8; ++q) b[q] = __builtin_amdgcn_raw_buffer_load_b128(rw, vw, so + q * 4 * 8192, 0);
      du = (ID_WS + (u32)(d - 2) * 16 + c) * 8 + cu * 2 + hh;
    } else {
      const u32 so = OFF_CH + unit_src(code, d, cu) * 8192;
#pragma unroll
      for (int q = 0; q < 8; ++q) b[q] = __builtin_amdgcn_raw_buffer_load_b128(rs, vl + (q & 3) * 1024, so + (q >> 2) * 4096, AUX_WT);
      du = code < 192 && d != 1 ? (ID_WS + (((((code - 64) >> 5) + 4) ^ (u32)d) - 2) * 16 + c) * 8 + cu * 2 + hh
                      : tile_dst(code, r, d) * 8 + hh * 4 + cu;
    }
    if (first && early && !confirm()) return false;
    const u32 me = k * 4;
    if (!(first && wv < NSEND)) {
      __builtin_amdgcn_s_waitcnt(0x0F70);
      if (first) { c0 = p.units[g][lane]; c1 = p.units[g][64 + lane]; }
      st[wv] = me + 1;
      while (true) {
        const u32 v = st[lane & 7];
        const u32 sending = (u32)__builtin_popcount((u32)__ballot((v & 3) == 2) & 0xffu);
        if (sending < NSEND && !__any((v & 3) == 1 && v < me)) break;
        __builtin_amdgcn_s_sleep(1);
      }
      st[wv] = me + 2;
    }
    if (k == gv[13] && cu < XWIN && lane == 0)
      __hip_atomic_fetch_add((GAS u32*)(ws + OFF_TK + (d & 3) * 256 + 128), (u32)(XWIN + 3 - cu) / 4, __ATOMIC_RELAXED, __HIP_MEMORY_SCOPE_AGENT);
    const u32 so = OFF_CH + du * 8192;
#pragma unroll
    for (int q = 0; q < 8; ++q)
      __builtin_amdgcn_raw_buffer_store_b128(b[q], rd, vl + (q & 3) * 1024, so + (q >> 2) * 4096, AUX_WT);
    asm volatile("s_waitcnt vmcnt(%0)" :: "i"(REL) : "memory");
    st[wv] = me;
    WAIT_VM(0);
    if (lane == 0) {
      char* fl = dws + OFF_FL + (du >> 3) * 64 + (du & 7) * 4;
      if (code >= 192 || d == 1) st_sys(fl, seq);
      else {
        const u32 pk = k - hh;
        const u32 old = __hip_atomic_fetch_or((LAS u32*)(st + 16 + (pk >> 5)), 1u << (pk & 31), __ATOMIC_RELAXED, __HIP_MEMORY_SCOPE_WORKGROUP);
        if ((old >> (pk & 31)) & 1) st_sys64(fl - hh * 4, ((u64)seq << 32) | seq);
      }
    }
    return true;
  };
  if (__builtin_expect(!early, 0) && !confirm()) return false;
  if (fixed && !send(true)) return false;
  if (!fixed || wv < NSEND) { c0 = p.units[g][lane]; c1 = p.units[g][64 + lane]; }
  __builtin_amdgcn_sched_barrier(0);
  {
    t0 = __builtin_amdgcn_s_memrealtime();
    const u32 q0 = unit_src(c0 >= 64 ? c0 : 64u, d, cu), q1 = unit_src(c1 >= 64 ? c1 : 64u, d, cu);
    f0 = OFF_FL + (q0 >> 3) * 64 + (q0 & 7) * 4; f1 = OFF_FL + (q1 >> 3) * 64 + (q1 & 7) * 4;
    rdy0 = __ballot(c0 < 64); rdy1 = __ballot(c1 < 64);
  }
  while (true) {
    bool have = false;
    while (!have) {
      const u32 v = st[lane & 15];
      auto rl = [&](int i) { return (u64)(u32)__builtin_amdgcn_readlane(v, i); };
      const u64 av0 = val0 & ~(rl(8) | rl(9) << 32), av1 = val1 & ~rl(10);
      if (!(av0 | av1)) break;
      rdy0 |= rl(12) | rl(13) << 32; rdy1 |= rl(14);
      const u32 head = av0 ? (u32)__builtin_ctzll(av0) : 64u + (u32)__builtin_ctzll(av1);
      const u64 ca0 = av0 & rdy0, ca1 = av1 & rdy1;
      k = ca0 ? (u32)__builtin_ctzll(ca0) : (ca1 ? 64u + (u32)__builtin_ctzll(ca1) : 128u);
      const u32 loaded = (u32)__builtin_popcount((u32)__ballot(v & 1) & 0xffu);
      u32 old = 0;
      if (k < 128 && (k == head || loaded < LOWM)) {
        if (lane == 0) old = __hip_atomic_fetch_or((LAS u32*)(st + 8 + (k >> 5)), 1u << (k & 31), __ATOMIC_RELAXED, __HIP_MEMORY_SCOPE_WORKGROUP);
        have = !((rfl(old) >> (k & 31)) & 1);
        if (have) code = k < 64 ? __builtin_amdgcn_readlane(c0, k) : __builtin_amdgcn_readlane(c1, k - 64);
        continue;
      }
      if (!rl(11) && lane == 0) old = __hip_atomic_exchange((LAS u32*)(st + 11), 1u, __ATOMIC_RELAXED, __HIP_MEMORY_SCOPE_WORKGROUP) ^ 1u;
      if (rfl(old)) {
        u32 a = 0, b = 0;
        asm volatile("" ::: "memory");
        if (((av0 & ~rdy0) >> lane) & 1) a = __builtin_amdgcn_raw_buffer_load_b32(rs, f0, 0, AUX_WT);
        if (((av1 & ~rdy1) >> lane) & 1) b = __builtin_amdgcn_raw_buffer_load_b32(rs, f1, 0, AUX_WT);
        rdy0 |= __ballot(a >= seq); rdy1 |= __ballot(b >= seq);
        if (give_up(ws, seq, t0, polls)) rdy0 = rdy1 = ~0ull;
        const u32 w = lane == 0 ? (u32)rdy0 : (lane == 1 ? (u32)(rdy0 >> 32) : (u32)rdy1);
        if (lane < 3) __hip_atomic_fetch_or((LAS u32*)(st + 12 + lane), w, __ATOMIC_RELAXED, __HIP_MEMORY_SCOPE_WORKGROUP);
        st[11] = 0;
      } else {
        __builtin_amdgcn_s_sleep(1);
      }
    }
    if (!have) break;
    send(false);
  }
  return true;
}

struct WStream { u32x4 d; u32 p, inc, v; };
struct XStream { u32x4 d; u32 p, inc, v0, v1; };
template <int OFF>
__device__ __forceinline__ void wload(WStream& o, u32 ldsbase) {
  asm volatile("s_add_u32 m0, %3, %4\n\ts_nop 0\n\tbuffer_load_dwordx4 %1, %2, %0 offen sc0 sc1 lds"
               :: "s"(o.p), "v"(o.v), "s"(o.d), "s"(ldsbase), "i"(OFF) : "memory", "scc");
  o.p += o.inc;
}
__device__ __forceinline__ void xload2(XStream& o, bf16x8& d0, bf16x8& d1) {
  asm volatile("buffer_load_dwordx4 %0, %3, %5, %2 offen sc0 sc1\n\tbuffer_load_dwordx4 %1, %4, %5, %2 offen sc0 sc1"
               : "+v"(d0), "+v"(d1) : "s"(o.p), "v"(o.v0), "v"(o.v1), "s"(o.d) : "memory");
  o.p += o.inc;
}
#define TIE_X(X) asm volatile("" : "+v"(X[0]), "+v"(X[1]), "+v"(X[2]), "+v"(X[3]) :: "memory")

__device__ __forceinline__ void gemm_tile(const Params& p, int j, int c, int rt, bool direct, u32 seq, char* smem, f32x4 (&res)[8]) {
  const int r = p.rank, tid = fresh_tid();
  int lane = tid & 63;
  const int wv8 = rfl(tid >> 6), grp = wv8 >> 2, wv = wv8 & 3;
  char* ws = p.wsd[0];
  const u32x4 rx = mk_desc(p.x, MM * KS * 2), rw = mk_desc(p.w, KS * NN * 2), rs = mk_desc(ws, WS_NEED);
  const int fr = lane & 15, fq = lane >> 4;
  const u32 krow = wv * 4 + fq, wch = (u32)(fr ^ fq ^ ((wv & 2) << 2));
  LAS char* lds = (LAS char*)smem + grp * 65536;
  const u32 ldsw = rfl((u32)(size_t)lds + wv * 1024);
  auto dm2 = [&](int pr) -> u32 { return (u32)(pr == 3 ? 0 : 2 + (((pr - 1) ^ c) & 1) * 2) * 16 * CHUNK; };
  const u32 wb_stg = rfl(OFF_CH + (ID_WS + grp * 16 + c) * CHUNK), xb_stg = rfl(OFF_CH + (ID_XS + grp * 16 + j * 2 + rt) * CHUNK);
  WStream wsx;
  XStream xsx;
  wsx.d = rw; wsx.p = rfl((u32)(((r >> 2) & 1) * 2048 + c * 128) * 2 + (u32)grp * 256 * 8192); wsx.inc = 16 * 8192;
  wsx.v = krow * 8192 + wch * 16;
  xsx.d = rx; xsx.p = rfl((u32)(blk_owner(r, j) * RS + blk_rb(r, j) * 256 + rt * 128) * 1024 + grp * 512); xsx.inc = 64;
  xsx.v0 = (u32)(wv * 32 + fr) * 1024 + fq * 16; xsx.v1 = xsx.v0 + 16384;
  auto wstage = [&](int pr, bool on) {
    wsx.d = rs; wsx.d[2] = on ? WS_NEED : 0u; wsx.p = wb_stg + dm2(pr); wsx.inc = 16 * 256;
    wsx.v = krow * 256 + wch * 16;
  };
  auto xstage = [&](int pr, bool on) {
    xsx.d = rs; xsx.d[2] = on ? WS_NEED : 0u; xsx.p = xb_stg + dm2(pr); xsx.inc = 1024;
    xsx.v0 = (u32)(wv * 16384 + lane * 16); xsx.v1 = xsx.v0 + 8192;
  };
  bf16x8 xr[2][4];
#pragma unroll
  for (int a = 0; a < 2; ++a)
#pragma unroll
    for (int b = 0; b < 4; ++b) xr[a][b] = bf16x8{};
  const int sg = lane >> 3, sdm2 = sg < 4 ? (sg ^ ((c & 1) * 2)) + 2 : (sg < 6 ? sg - 4 : 0);
  const u32 pgo = OFF_FL + ((lane & 4) ? ID_WS + sdm2 * 16 + c : ID_XS + sdm2 * 16 + j * 2 + rt) * 64 + (lane & 3) * 8;
  u64 pv = 0;
  auto pairs_ready = [&](u64 v) -> u32 {
    const u64 b = __ballot((u32)v >= seq && (u32)(v >> 32) >= seq);
    return ((b & 0xffff) == 0xffff ? 1u : 0u) | ((b >> 16 & 0xffff) == 0xffff ? 2u : 0u) | ((b >> 32 & 0xffff) == 0xffff ? 4u : 0u);
  };
  volatile LAS u32* kn = (volatile LAS u32*)((LAS char*)smem + LDS_CTL);

  const int qq = fr >> 2, pp = lane & 3;
  const u32 wsw = (u32)(qq | ((fq & 1) << 3));
  u32 wa[4];
#pragma unroll
  for (int t = 0; t < 4; ++t) {
    const u32 tt = (u32)(t ^ (pp & 1));
    wa[t] = (u32)(8 * fq + qq) * 256 + ((((u32)(pp * 2) + (tt >> 1)) ^ wsw) * 16) + (tt & 1) * 8;
  }
  f32x4 acc[2][8];
#pragma unroll
  for (int a = 0; a < 2; ++a)
#pragma unroll
    for (int b = 0; b < 8; ++b) acc[a][b] = f32x4{0.f, 0.f, 0.f, 0.f};
  bf16x8 wf[2][8];
  auto wread2 = [&](int st, int i) {
#pragma unroll
    for (int f = 2 * i; f < 2 * i + 2; ++f) {
      const int kk = f >> 3, t = f & 7;
      const LAS char* a = lds + (wa[t & 3] ^ ((t >> 2) * 128)) + st * 16384 + kk * 8192;
      const s16x4 lo = __builtin_amdgcn_ds_read_tr16_b64_v4i16((LAS s16x4*)a);
      const s16x4 hi = __builtin_amdgcn_ds_read_tr16_b64_v4i16((LAS s16x4*)(a + 1024));
      wf[kk][t] = __builtin_bit_cast(bf16x8, __builtin_shufflevector(lo, hi, 0, 1, 2, 3, 4, 5, 6, 7));
    }
  };
  auto lphase = [&](int st, bf16x8* xn) {
#pragma unroll
    for (int i = 0; i < 2; ++i) {
#pragma unroll
      for (int q = 0; q < 4; ++q) wread2(st, 4 * i + q);
      __builtin_amdgcn_sched_barrier(0);
      xload2(xsx, xn[i], xn[2 + i]);
    }
    WAIT_VM(8); WAIT_LGKM0(); BARRIER();
  };
  auto mma4 = [&](const bf16x8* x, int i) {
#pragma unroll
    for (int f = 4 * i; f < 4 * i + 4; ++f) {
      const int kk = f >> 3, t = f & 7;
      acc[0][t] = __builtin_amdgcn_mfma_f32_16x16x32_bf16(wf[kk][t], x[kk], acc[0][t], 0, 0, 0);
      acc[1][t] = __builtin_amdgcn_mfma_f32_16x16x32_bf16(wf[kk][t], x[2 + kk], acc[1][t], 0, 0, 0);
    }
    const int t0 = (4 * i) & 7;
    asm volatile("" : "+v"(acc[0][t0]), "+v"(acc[0][t0 + 1]), "+v"(acc[0][t0 + 2]), "+v"(acc[0][t0 + 3]),
                      "+v"(acc[1][t0]), "+v"(acc[1][t0 + 1]), "+v"(acc[1][t0 + 2]), "+v"(acc[1][t0 + 3]));
    __builtin_amdgcn_sched_barrier(0);
  };
#define MPHASE(X, LST) do {                                                    \
    TIE_X(X);                                                                  \
    __builtin_amdgcn_sched_barrier(0);                                         \
    __builtin_amdgcn_s_setprio(1);                                             \
    wload<(LST) * 16384>(wsx, ldsw); mma4(X, 0);                               \
    wload<(LST) * 16384 + 4096>(wsx, ldsw); mma4(X, 1);                        \
    wload<(LST) * 16384 + 8192>(wsx, ldsw); mma4(X, 2);                        \
    wload<(LST) * 16384 + 12288>(wsx, ldsw); mma4(X, 3);                       \
    __builtin_amdgcn_s_setprio(0);                                             \
    WAIT_VM(8);                                                                \
  } while (0)

  u32 known = 0, left = 7, polls = 0;
  const u64 t0 = __builtin_amdgcn_s_memrealtime();
  __builtin_amdgcn_s_waitcnt(0xC07F);
  bool restart = true;
  for (int it = 0; it < 4; ++it) {
    if (restart) {
      asm volatile("s_nop 4" ::: "memory");
      wload<0>(wsx, ldsw); wload<4096>(wsx, ldsw); wload<8192>(wsx, ldsw); wload<12288>(wsx, ldsw);
      xload2(xsx, xr[0][0], xr[0][2]); xload2(xsx, xr[0][1], xr[0][3]);
      wload<16384>(wsx, ldsw); wload<20480>(wsx, ldsw); wload<24576>(wsx, ldsw); wload<28672>(wsx, ldsw);
      if (grp == 1) BARRIER();
      WAIT_VM(8);
      BARRIER();
    }
    const bool ask = left && !(known & left);
    lphase(0, xr[1]);
    MPHASE(xr[0], 2); BARRIER();
    if (ask && wv8 == 0) asm volatile("buffer_load_dwordx2 %0, %1, %2, 0 offen sc0 sc1" : "+v"(pv) : "v"(pgo), "s"(rs) : "memory");
    lphase(1, xr[0]);
    MPHASE(xr[1], 3);
    asm volatile("" : "+v"(pv) :: "memory");
    if (ask && wv8 == 0) { *kn = pairs_ready(pv); WAIT_LGKM0(); }
    BARRIER();
    if (ask) known = rfl(*kn);
    const bool go = known & left;
    int pr = __builtin_ctz((known & left) | 8u) + 1;
    wstage(pr, go);
    lphase(2, xr[1]);
    xstage(pr, go);
    MPHASE(xr[0], 0); BARRIER();
    lphase(3, xr[0]);
    MPHASE(xr[1], 1); BARRIER();
    restart = !go;
    if (!go) {
      if (left) {
        if (wv8 == 0) {
          u32 k;
          while (true) {
            k = pairs_ready(__hip_atomic_load((const u64*)(ws + pgo), __ATOMIC_RELAXED, __HIP_MEMORY_SCOPE_SYSTEM));
            if (k & left) break;
            if (give_up(ws, seq, t0, polls)) { k = 7; break; }
            __builtin_amdgcn_s_sleep(2);
          }
          *kn = k;
          WAIT_LGKM0();
        }
        if (grp == 0) BARRIER();
        known = rfl(*kn);
        pr = __builtin_ctz(known & left) + 1;
        wstage(pr, true); xstage(pr, true);
      }
    }
    left &= ~(1u << (pr - 1));
  }
  WAIT_VM(0);
  TIE_X(xr[0]);
  lane = fresh_tid() & 63;
  const bool gated = direct && j >= 4;
  const bool pres = j >= 4 && ((PRE >> (c * 2 + rt)) & 1) && ((j ^ c) & 1) == 0;
  u32 tk = 0;
  auto ticket = [&]() {
    if (wv8 == 0 && lane == 0)
      tk = __hip_atomic_fetch_add((GAS u32*)(ws + OFF_TK + (j & 3) * 256), 1u, __ATOMIC_RELAXED, __HIP_MEMORY_SCOPE_AGENT);
  };
  if (gated && !pres) ticket();
  LAS f32x4* ex = (LAS f32x4*)((LAS char*)smem + 32768);
  if (grp == 0) {
#pragma unroll
    for (int t = 0; t < 8; ++t) { ex[(wv * 8 + t) * 64 + lane] = acc[1][t]; res[t] = acc[0][t]; }
    BARRIER();
  } else {
#pragma unroll
    for (int t = 0; t < 8; ++t) { ex[4096 + (wv * 8 + t) * 64 + lane] = acc[0][t]; res[t] = acc[1][t]; }
  }
  WAIT_LGKM0();
  BARRIER();
#pragma unroll
  for (int t = 0; t < 8; ++t) res[t] += ex[(grp ^ 1) * 4096 + (wv * 8 + t) * 64 + lane];
  if ((j & 6) == 2) return;
  if (pres) {
    const u32 tr = ID_TR + (u32)(((j >> 1) & 1) * 16 + c);
    if (wv8 == 0) {
      while (!__all(ld_sys(ws + OFF_FL + tr * 64 + (rt * 4 + (lane & 3)) * 4) >= seq)) {
        if (give_up(ws, seq, t0, polls)) break;
        __builtin_amdgcn_s_sleep(4);
      }
    }
    if (gated) ticket();
    BARRIER();
    const rsrc_t rl = mk_rsrc(ws, WS_NEED);
#pragma unroll
    for (int hf = 0; hf < 4; ++hf) {
      const u32x4 a = __builtin_amdgcn_raw_buffer_load_b128(rl, lane * 16 + hf * 1024, OFF_CH + tr * CHUNK + rt * 32768 + wv * 8192 + grp * 4096, AUX_WT);
      res[2 * hf] += f32x4{bf_lo(a[0]), bf_hi(a[0]), bf_lo(a[1]), bf_hi(a[1])};
      res[2 * hf + 1] += f32x4{bf_lo(a[2]), bf_hi(a[2]), bf_lo(a[3]), bf_hi(a[3])};
    }
  }
  char* ows = direct ? p.wsd[(blk_owner(r, j) ^ r) & (NR - 1)] : ws;
  const u32 id = direct ? ps_id(r, j, c) : ID_OB + (u32)j * 16 + c;
  const rsrc_t rd = mk_rsrc(ows, WS_NEED);
  const u32 so = OFF_CH + id * CHUNK + rt * 32768 + wv * 8192 + grp * 4096;
  GAS u32* cr = (GAS u32*)(ws + OFF_TK + (j & 3) * 256 + 128);
  u32x4 pk[4];
#pragma unroll
  for (int hf = 0; hf < 4; ++hf) {
    const f32x4 a = res[2 * hf], b = res[2 * hf + 1];
    pk[hf] = u32x4{cvt_pk_bf16(a[0], a[1]), cvt_pk_bf16(a[2], a[3]), cvt_pk_bf16(b[0], b[1]), cvt_pk_bf16(b[2], b[3])};
  }
  if (gated) {
    asm volatile("" :: "v"(pk[0]), "v"(pk[1]), "v"(pk[2]), "v"(pk[3]));
    if (wv8 == 0) {
      kn[1] = 0;
      WAIT_LGKM0();
      const u32 need = rfl(tk) + (seq - 1) * XWIN;
      while (rfl(__hip_atomic_load(cr, __ATOMIC_RELAXED, __HIP_MEMORY_SCOPE_AGENT)) <= need) {
        if (give_up(ws, seq, t0, polls)) break;
        __builtin_amdgcn_s_sleep(1);
      }
    }
    BARRIER();
  }
#pragma unroll
  for (int hf = 0; hf < 4; ++hf) __builtin_amdgcn_raw_buffer_store_b128(pk[hf], rd, lane * 16 + hf * 1024, so, AUX_WT);
  WAIT_VM(0);
  if (gated && lane == 0) {
    u32 n;
    asm volatile("ds_add_rtn_u32 %0, %1, %2 offset:4\n\ts_waitcnt lgkmcnt(0)" : "=&v"(n) : "v"(kn), "v"(1u) : "memory");
    if (n == XRELN) __hip_atomic_fetch_add(cr, 1u, __ATOMIC_RELAXED, __HIP_MEMORY_SCOPE_AGENT);
  }
  WAIT_LGKM0();
  BARRIER();
  if (wv8 == 0 && lane < 4) st_sys(ows + OFF_FL + id * 64 + (rt * 4 + lane) * 4, seq);
}

__device__ __forceinline__ u32 wave_max(u32 v) {
  v = max(v, (u32)__builtin_amdgcn_update_dpp(0, (int)v, 0x111, 0xf, 0xf, true));
  v = max(v, (u32)__builtin_amdgcn_update_dpp(0, (int)v, 0x112, 0xf, 0xf, true));
  v = max(v, (u32)__builtin_amdgcn_update_dpp(0, (int)v, 0x114, 0xf, 0xf, true));
  v = max(v, (u32)__builtin_amdgcn_update_dpp(0, (int)v, 0x118, 0xf, 0xf, true));
  const u32 a = __builtin_amdgcn_readlane(v, 15), b = __builtin_amdgcn_readlane(v, 31);
  const u32 c = __builtin_amdgcn_readlane(v, 47), d = __builtin_amdgcn_readlane(v, 63);
  return max(max(a, b), max(c, d));
}

__device__ __forceinline__ void reduce_tile(const Params& p, int type, int rb, int c, int rt, u32 seq, char* smem, f32x4 (&y)[8]) {
  const int r = p.rank, tid = fresh_tid(), lane = tid & 63;
  const int wv = rfl(tid >> 6), blk = (wv & 3) * 2 + (wv >> 2);
  char* ws = p.wsd[0];
  char* peer = p.wsd[lane & (NR - 1)];
  const rsrc_t rs = mk_rsrc(ws, WS_NEED);
  const u32 id0 = ID_PS + (u32)(((type * 2 + rb) * 16 + c) * 2);
  const u64 t0 = __builtin_amdgcn_s_memrealtime();
  volatile LAS u32* red = (volatile LAS u32*)((LAS char*)smem + LDS_CTL + 64);
  u32 polls = 0;
  const char* fp = ws + OFF_FL + (id0 + ((lane >> 2) & 1)) * 64 + (rt * 4 + (lane & 3)) * 4;
  const u32 so = OFF_CH + id0 * CHUNK + rt * 32768 + blk * 4096;
  u32 todo = type == 0 ? 2u >> (r & 1) : ((PRE >> (c * 2 + rt)) & 1 ? 1u << ((r ^ c) & 1) : 3u);
  if (type) {
#pragma unroll
    for (int t = 0; t < 8; ++t) y[t] = f32x4{0.f, 0.f, 0.f, 0.f};
  }
  auto add = [&](const u32x4* a) {
#pragma unroll
    for (int i = 0; i < 4; ++i)
#pragma unroll
      for (int e = 0; e < 4; ++e) {
        y[2 * i + (e >> 1)][(e & 1) * 2] += bf_lo(a[i][e]);
        y[2 * i + (e >> 1)][(e & 1) * 2 + 1] += bf_hi(a[i][e]);
      }
  };
  const u64* vp = (const u64*)(ws + OFF_VD + (blockIdx.x & 15) * 128);
  LAS u32* mx = (LAS u32*)red;
  u64 vd = 0;
  for (u32 round = type ? 0u : 1u; todo; ++round) {
    if (wv == 0) {
      u32 m = 0;
      while (round) {
        const u32 b = (u32)__ballot(ld_sys(fp) >= seq);
        m = (((b & 15) == 15 ? 1u : 0u) | ((b & 0xf0) == 0xf0 ? 2u : 0u)) & todo;
        if (m) break;
        if (give_up(ws, seq, t0, polls)) { m = todo; break; }
        __builtin_amdgcn_s_sleep(FSLEEP);
      }
      red[16 + round * 4 + (lane & 3)] = lane & 3 ? 0u : m;
      WAIT_LGKM0();
    }
    BARRIER();
    const u32 m = rfl(red[16 + round * 4]);
    const rsrc_t ra = mk_rsrc(ws, m ? WS_NEED : 0u);
    u32x4 a[4];
#pragma unroll
    for (int i = 0; i < 4; ++i) a[i] = __builtin_amdgcn_raw_buffer_load_b128(ra, lane * 16 + i * 1024, so + (m & 1 ? 0u : CHUNK), AUX_WT);
    if (m == 3) {
      u32x4 b[4];
#pragma unroll
      for (int i = 0; i < 4; ++i) b[i] = __builtin_amdgcn_raw_buffer_load_b128(rs, lane * 16 + i * 1024, so + CHUNK, AUX_WT);
      add(b);
    }
    if (wv == 0) vd = __hip_atomic_load(vp, __ATOMIC_RELAXED, __HIP_MEMORY_SCOPE_SYSTEM);
    char* gp = peer;
    asm volatile("" : "+v"(gp));
    gp += OFF_GG + (r * 128 + type * 64 + c * 4 + rb * 2 + rt) * 8;
    add(a);
    todo &= ~m;
    float mf = 0.f;
#pragma unroll
    for (int t = 0; t < 8; ++t) mf = fmaxf(fmaxf(mf, fmaxf(fabsf(y[t][0]), fabsf(y[t][1]))), fmaxf(fabsf(y[t][2]), fabsf(y[t][3])));
    const u32 wm = wave_max(__builtin_bit_cast(u32, mf));
    mx = (LAS u32*)red + 17 + round * 4;
    u32 tk = 0, all = 0;
    if (lane == 0)
    asm volatile("ds_max_u32 %2, %3\n\tds_add_rtn_u32 %0, %2, %4 offset:4\n\tds_read_b32 %1, %2\n\ts_waitcnt lgkmcnt(0)"
                 : "=&v"(tk), "=&v"(all) : "v"(mx), "v"(wm), "v"(1u) : "memory");
    if (!todo && rfl(tk) == 7 && lane < NR) st_sys64(gp, ((u64)seq << 32) | rfl(all));
  }
  const int fr = lane & 15, fq = lane >> 4;
  const rsrc_t ro = mk_rsrc(p.out, RS * NN);
  const u32 op = (u32)(rb * 256 + rt * 128 + blk * 16 + fr) * NN + (((r >> 2) & 1) ^ type) * 2048 + c * 128 + fq * 16;
  u32 used = ~1u;
  for (int it = 0;; ++it) {
    if (wv == 0) {
      if (it) {
        const u64 last = vd;
        while (true) {
          vd = __hip_atomic_load(vp, __ATOMIC_RELAXED, __HIP_MEMORY_SCOPE_SYSTEM);
          const u32 tm = ld_sys(ws + OFF_TMO);
          const bool news = __all(vd != last);
          if (rfl(tm) == seq || (!news && give_up(ws, seq, t0, polls))) { vd = ((u64)(2 * seq + 1) << 32) | ~0u; break; }
          if (news) break;
          if (rfl((u32)(vd >> 32)) < 2 * seq) __builtin_amdgcn_s_sleep(8); else __builtin_amdgcn_s_sleep(GSLEEP);
        }
      }
      red[8 + (it & 1) * 2] = (u32)vd;
      red[9 + (it & 1) * 2] = (u32)(vd >> 32);
    }
    WAIT_LGKM0();
    BARRIER();
    const u32 own = rfl(*(volatile LAS u32*)mx);
    const u32 tag = rfl(red[9 + (it & 1) * 2]), gmb = max(own, rfl(red[8 + (it & 1) * 2]));
    if (tag >= 2 * seq && gmb != used) {
      used = gmb;
      const bool bad = gmb == ~0u;
      const float gm = as_f(gmb), inv = gm > 0.f ? 127.0f / gm : 0.f;
#pragma unroll
      for (int mi = 0; mi < 2; ++mi) {
        u32 d[4];
#pragma unroll
        for (int t = 0; t < 4; ++t) {
          u32 q[4];
#pragma unroll
          for (int e = 0; e < 4; ++e)
            q[e] = __builtin_bit_cast(u32, __builtin_amdgcn_fmed3f(y[mi * 4 + t][e] * inv, -127.f, 127.f) + 12582912.f);
          const u32 pk = __builtin_amdgcn_perm(__builtin_amdgcn_perm(q[3], q[2], 0x0c0c0400u), __builtin_amdgcn_perm(q[1], q[0], 0x0c0c0400u), 0x05040100u);
          d[t] = bad ? 0x80808080u : pk;
        }
        const bool odd = fq & 1;
        __builtin_amdgcn_raw_buffer_store_b128(u32x4{odd ? d[1] : d[0], odd ? d[0] : d[1], odd ? d[3] : d[2], odd ? d[2] : d[3]},
                                               ro, op + mi * 64, 0, OAUX);
      }
    }
    if (tag > 2 * seq) break;
  }
}

__device__ __forceinline__ void master_role(const Params& p, u32 seq, char* smem) {
  const int tid = fresh_tid(), lane = tid & 63;
  char* ws = p.wsd[0];
  const rsrc_t rs = mk_rsrc(ws, WS_NEED);
  LAS u32* sh = (LAS u32*)((LAS char*)smem + LDS_CTL + 64);
  if (tid < 4) sh[tid] = 0;
  WAIT_LGKM0();
  BARRIER();
  const u64 t0 = __builtin_amdgcn_s_memrealtime();
  u32 polls = 0;
  bool half = false;
  char* vp = ws + OFF_VD + (lane & 15) * 128;
  u32 seen = 65;
  while (true) {
    u32x4 g;
    u32 miss;
    bool bad = false;
    while (true) {
      asm volatile("" ::: "memory");
      g = __builtin_amdgcn_raw_buffer_load_b128(rs, tid * 16, OFF_GG, AUX_WT);
      miss = (u32)__builtin_popcountll(__ballot(min(g[1], g[3]) < seq));
      if (miss < seen) break;
      if (give_up(ws, seq, t0, polls)) { bad = true; break; }
      if (miss > GNEAR) __builtin_amdgcn_s_sleep(8);
    }
    seen = miss;
    u32 m = wave_max(max(g[1] >= seq ? g[0] : 0u, g[3] >= seq ? g[2] : 0u));
    if (bad) { m = ~0u; miss = 0; }
    u32 o = 0, nd = 0, nh = 0;
    if (lane == 0)
      asm volatile("ds_max_u32 %3, %4\n\tds_add_rtn_u32 %0, %3, %5 offset:4\n\tds_add_rtn_u32 %1, %3, %6 offset:8\n\t"
                   "ds_read_b32 %2, %3\n\ts_waitcnt lgkmcnt(0)"
                   : "=&v"(nd), "=&v"(nh), "=&v"(o) : "v"(sh), "v"(m), "v"(miss ? 0u : 1u), "v"(!half && miss <= GNEAR ? 1u : 0u) : "memory");
    half = half || miss <= GNEAR;
    if (!miss) {
      if (rfl(nd) == 7) {
        if (lane < 16) st_sys64(vp, ((u64)(2 * seq + 1) << 32) | rfl(o));
      }
      break;
    }
    if (rfl(nh) >= 4) {
      u32 prev = 0;
      if (lane == 0) prev = __hip_atomic_fetch_max(sh + 3, o, __ATOMIC_RELAXED, __HIP_MEMORY_SCOPE_WORKGROUP);
      if (rfl(prev) < rfl(o)) {
        if (lane < 16) st_sys64(vp, ((u64)(2 * seq) << 32) | rfl(o));
        WAIT_VM(0);
      }
    }
  }
}

__global__ void __launch_bounds__(NT, 2) fused_kernel(Params p) {
  extern __shared__ __attribute__((aligned(16))) char smem[];
  u32 xcc, seq = 0;
  u64 pc0;
  asm volatile("s_getpc_b64 %1\n\ts_getreg_b32 %0, hwreg(HW_REG_XCC_ID, 0, 4)" : "=s"(xcc), "=s"(pc0));
  u32 slot = (blockIdx.x & ~7u) | (xcc & 7u);
  auto is_pusher = [](u32 sl) { return (sl & 7) != 0 && sl < NPC * 8; };
  if (__builtin_expect(is_pusher(slot), 1)) {
    if (push_role<true>(p, slot, seq, smem)) return;
  } else {
    if (slot >= 128 && slot < 240) {
      const u32x16 gv = p.grp[slot & 7];
      if (gv[11] == 1) {
        const u32 i = ((slot >> 3) - 16) * NT + threadIdx.x, k = i - 4096;
        const char* a = i < 4096 ? (const char*)((u64)gv[0] | (u64)gv[1] << 32) + (gv[10] ^ 1) * RS * 1024 + i * 128
                                 : (const char*)((u64)gv[2] | (u64)gv[3] << 32) + (k / 6) * 8192 + ((gv[10] >> 2) & 1) * 4096 + (k % 6) * 128;
        const u32 v = *(const GAS u32*)a;
        asm volatile("" :: "v"(v));
      }
    }
    if (slot >= NWG - 8) {
      const u32x16 gv = p.grp[slot & 7];
      const int lane = threadIdx.x & 63, wv = rfl(threadIdx.x >> 6);
      const char* dws = (const char*)((u64)gv[6] | (u64)gv[7] << 32);
      const u32 dm2 = (gv[11] - 2) & 7;
      const char* a = dws;
      if (wv == 0) a = dws + (lane == 0 ? OFF_FL : OFF_CH + ((lane & 1 ? ID_XS : ID_WS) + (dm2 < 6 ? dm2 : 0) * 16 + (lane & 2) * 7) * CHUNK);
      else {
        for (int i = 0; i < PRETOUCH; ++i) __builtin_amdgcn_s_sleep(16);
        if (wv < 3) a = (wv == 1 ? dws : p.wsd[0]) + (u32)(lane < 62 ? lane : 0) * 524288;
        else if (wv == 3) a = lane < 8 ? p.wsd[lane] + OFF_GG : p.out + (lane & 7) * 262144;
      }
      if (wv < 4 && (wv || lane < 5)) { const u32 v = ld_sys(a); asm volatile("" :: "v"(v)); }
    } else if (slot >= NWG - 16 && threadIdx.x < CODE_LINES) {
      const u32 v = *(const GAS u32*)((pc0 & ~127ull) + threadIdx.x * 128);
      asm volatile("" :: "v"(v));
    }
    seq = claim_resolve(p.wsd[0], claim_issue(p.wsd[0], slot), slot, smem);
  }
  if (__builtin_expect(is_pusher(slot), 0)) {
    BARRIER();
    push_role<false>(p, slot, seq, smem);
    return;
  }
  const u32 plan = p.plan[slot];
  for (int i = plan >> 28; i > 0; --i) __builtin_amdgcn_s_sleep(32);
  f32x4 y[8];
  for (int it = 0; it < 2; ++it) {
    const u32 t = rfl(plan >> (it * 8));
    gemm_tile(p, (t >> 5) & 7, (t >> 1) & 15, t & 1, (plan >> (26 + it)) & 1, seq, smem, y);
    if (!((plan >> 24) & 1)) break;
  }
  if ((plan >> 23) & 1) master_role(p, seq, smem);
  if ((plan >> 25) & 1) reduce_tile(p, (plan >> 22) & 1, (plan >> 17) & 1, (plan >> 18) & 15, (plan >> 16) & 1, seq, smem, y);
}

const char* const SCHED1 = "p3q3p4q4p5q5p6q6s0s1p7q7t0t1p8q8p9q9paqapbqb";
const char* const SCHED23 = "x0w0x1w1w2x2x3w3w4w5x8xaw6xcw7xew8x9w9xbrawaxdrbwbxfrcwcrdwdrewerfwfx4x5x6x7qaqbpapb";
const char* const SCHED45 = "x0w0x1w1w2x2x3w3w4w5x8xaw6xcw7wawbxew8x9w9xbwcwdxdxfwewfx4x5x6x7p0p1";
const char* const SCHED67 = SCHED45;

void build_plan(Params& p, const int* link) {
  auto tile = [](int j, int c, int rt) { return (u32)(j * 32 + c * 2 + rt); };
  int own_next = 0, red_next = 0;
  auto take_red = [&]() { const int u = 63 - red_next++; return (1u << 25) | ((64u + (u32)u) << 16); };
  auto own_red = [](u32 t) { return (1u << 25) | ((((t >> 1) & 15) * 4 + ((t >> 5) & 1) * 2 + (t & 1)) << 16); };
  auto second_own = [&]() {
    const u32 t2 = tile(2 + ((own_next >> 1) & 1), own_next >> 2, own_next & 1);
    ++own_next;
    return t2 << 8 | 1u << 24 | 1u << 27 | own_red(t2);
  };
  for (int b = 0; b < NWG; ++b) p.plan[b] = 0;
  for (int pass = 0; pass < 3; ++pass)
    for (int b = 0; b < NWG; ++b) {
      const int j = link[b & 7], wi = b >> 3, wk = j == 0 ? wi : wi - NPC;
      if (wk < 0) continue;
      if (pass == 1 && j == 0) {
        p.plan[b] = tile((wk >> 1) & 1, NEARLY + (wk >> 2), wk & 1) | (u32)STARTDELAY << 28;
        if (NEARLY + (wk >> 2) < 7) p.plan[b] |= second_own(); else if (NEARLY + (wk >> 2) < 11) p.plan[b] |= take_red();
      } else if (pass == 0 && j == 1) {
        const int c = wk < 4 * NEARLY ? wk >> 2 : 9 + (wk >> 2);
        p.plan[b] = tile((wk >> 1) & 1, c, wk & 1) | 1u << 26 | (c < 7 ? second_own() : 0u) | (u32)(c < 3 ? 0 : STARTDELAY) << 28;
      } else if (pass == 2 && j >= 4) {
        p.plan[b] = tile(j, 2 + (wk >> 1), wk & 1) | (u32)STARTDELAY << 28;
        const bool ship = (((unsigned)PRE >> (4 + wk)) & 1) && ((j ^ (wk >> 1)) & 1);
        if (2 + (wk >> 1) >= XDIR && !ship) p.plan[b] |= 1u << 26;
        if (wk < 8) p.plan[b] |= take_red();
        if (wk == 8 && j == 4) p.plan[b] |= 1u << 23;
      } else if (pass == 0 && (j == 2 || j == 3)) {
        const int q = wk * 2 + (j & 1);
        if (q < 16) p.plan[b] = tile(4 + (q & 3), q >> 3, (q >> 2) & 1) | take_red();
        else if (q < 20) p.plan[b] = tile(q & 1, NEARLY + 8, (q >> 1) & 1);
        else {
          const int o = 28 + q - 20;
          const u32 t = tile(2 + (o & 1), o >> 2, (o >> 1) & 1);
          p.plan[b] = t | 1u << 26 | own_red(t);
        }
        p.plan[b] |= (u32)STARTDELAY << 28;
      }
    }
}

void build_jobs(Params& p, const int* link) {
  const char* sc[4] = {SCHED1, SCHED23, SCHED45, SCHED67};
  auto hex = [](char ch) { return ch <= '9' ? ch - '0' : ch - 'a' + 10; };
  unsigned relayed = 0, off1 = YDIR;
  for (const char* t = sc[1]; *t; t += 2) {
    if (t[0] == 'r') relayed |= 1u << hex(t[1]);
    if (t[0] == 'p') off1 |= 1u << hex(t[1]);
  }
  for (int g = 0; g < NR; ++g) {
    const int d = link[g];
    unsigned char* q = p.units[g];
    memset(q, 0, sizeof p.units[g]);
    int n = 0;
    for (const char* t = d == 0 ? "" : sc[d == 1 ? 0 : (d < 4 ? 1 : (d < 6 ? 2 : 3))]; *t && n < MAXU; t += 2) {
      const int v = hex(t[1]);
      int code = -1;
      if (t[0] == 'x') code = v * 2;
      else if (t[0] == 'w') { if (d < 4 || !((relayed >> v) & 1) || ((d >> 1) & 1) == (v & 1)) code = 32 + v * 2; }
      else if (t[0] == 'r') code = 64 + ((v & 1) * 2 + ((p.rank ^ d) & 1)) * 32 + v * 2;
      else if (t[0] == 's' || t[0] == 't') code = 64 + (t[0] == 't' ? 32 : 0) + v * 2;
      else code = 192 + (t[0] == 'q' ? 32 : 0) + v * 2;
      if (code >= 192 && (d == 1 ? (off1 >> v) & 1 : d >= 4 && v >= XDIR)) code = -1;
      for (int hh = 0; code >= 0 && hh < 2; ++hh) {
        const bool pre = ((unsigned)PRE >> ((code + hh) & 31)) & 1;
        if (d == 1 && code < 192 ? !pre : code >= 192 && d >= 4 && pre && ((d ^ v) & 1)) continue;
        q[n++] = (unsigned char)(code + hh);
      }
    }
    Grp ga = {p.x, p.w, p.wsd[0], p.wsd[d], 0, (u32)p.rank, (u32)d, (u32)(n < MAXU ? n : MAXU), ~0u, {0, 0}};
    for (int u = 0; u < n && d >= 4; ++u) if (q[u] < 64) ga.open = u - XEARLY;
    memcpy(&ga.first8, q, 8);
    memcpy(&p.grp[g], &ga, 64);
  }
}

}

extern "C" void kernel_launch(int rank, int nranks, ncclComm_t comm,
                              hipStream_t stream,
                              void* const* d_in, const int* in_sizes, int n_in,
                              void* d_out, int out_size,
                              const int* in_dtypes, int out_dtype,
                              void* const* d_out_all,
                              void* const* d_ws_all, size_t ws_bytes) {
  (void)comm; (void)in_dtypes; (void)out_dtype; (void)d_out_all;
  if (nranks != NR || n_in != 2 || in_sizes[0] != MM * KS || in_sizes[1] != KS * NN || out_size != RS * NN ||
      ws_bytes < WS_NEED || (unsigned)rank >= (unsigned)NR) {
    fprintf(stderr, "unexpected problem shape\n");
    return;
  }
  HIP_CHECK(hipFuncSetAttribute((const void*)fused_kernel, hipFuncAttributeMaxDynamicSharedMemorySize, LDS_BYTES));
  Params p;
  p.x = (const char*)d_in[0];
  p.w = (const char*)d_in[1];
  p.out = (char*)d_out;
  for (int i = 0; i < NR; ++i) p.wsd[i] = (char*)d_ws_all[rank ^ i];
  p.rank = rank;
  static const unsigned char near03[4] = {0x4c, 0x70, 0x16, 0x2a};
  const unsigned m = near03[rank < 4 ? rank : 7 - rank];
  int link[NR] = {0};
  for (int d = 1, a = 1, b = 4; d < NR; ++d) link[(m >> d) & 1 ? a++ : b++] = d;
  build_jobs(p, link);
  build_plan(p, link);
  hipLaunchKernelGGL(fused_kernel, dim3(NWG), dim3(NT), LDS_BYTES, stream, p);
  HIP_CHECK(hipGetLastError());
}
```
